```python
import numpy as np

kernel_name = "gpu_ag_gemm_m4096_k4096_n32768_bf16_none"


ABSMAX_RTOL = 0.008

COMPUTE_DTYPE = "bfloat16"
EPILOGUE = "none"
IN_QUANT = None
OUT_QUANT = None
IN_DTYPES = {"x": "bfloat16", "w_mat": "bfloat16"}
WORLD_SIZE = 8

def _to_bf16(x):
    u = np.ascontiguousarray(x, dtype=np.float32).view(np.uint32)
    return ((u + 0x7FFF + ((u >> 16) & 1)) & 0xFFFF0000).view(np.float32)

def setup_inputs():

    x = _to_bf16(np.random.randn(4096, 4096).astype(np.float32))
    w_mat = _to_bf16((np.random.randn(4096, 32768) / np.sqrt(4096)).astype(np.float32))
    return {"x": x, "w_mat": w_mat}

def _epilogue(y):
    return y

def reference(x, w_mat):
    y = (x @ w_mat).astype(np.float32)
    return _epilogue(y)
```

```cpp
#include <hip/hip_runtime.h>
#include <rccl/rccl.h>
#include <cstdio>
#include <type_traits>

#define HIP_CHECK(expr) do { hipError_t _e = (expr); if (_e != hipSuccess) { \
    fprintf(stderr, "HIP error %s at %s:%d\n", hipGetErrorString(_e), __FILE__, __LINE__); \
    return; } } while (0)

typedef unsigned short u16;
typedef short s16x4 __attribute__((ext_vector_type(4)));
typedef short s16x8 __attribute__((ext_vector_type(8)));
typedef float f32x4 __attribute__((ext_vector_type(4)));
typedef float f32x2 __attribute__((ext_vector_type(2)));
typedef __bf16 bf16x2 __attribute__((ext_vector_type(2)));
typedef unsigned u32x4 __attribute__((ext_vector_type(4)));
typedef unsigned u32x2 __attribute__((ext_vector_type(2)));
typedef int i32x4 __attribute__((ext_vector_type(4)));
#define LAS __attribute__((address_space(3)))

constexpr int NR = 8;
constexpr int M = 4096, N = 4096, K = 4096;
constexpr int SLOT = 16384, NSW = 4, NSX = 6;
constexpr int NJ = K / 64;
constexpr int LDS_BYTES = (NSW + NSX) * SLOT;
constexpr int FIRST_KT = 4;
constexpr int PF_PRE = 4;
constexpr size_t WS_CNT = 0;
constexpr size_t WS_CLAIM = 2048;
constexpr size_t WS_TMO = 4096;
constexpr size_t WS_INV = 4096 + 256;
constexpr size_t WS_NAN = 16384;
constexpr size_t WS_FLAGS = 131072;
constexpr size_t WS_FIRST = 262144;
constexpr size_t WS_DATA = 1u << 20;
constexpr size_t REGION = (size_t)256 * K * 2;
constexpr size_t WS_HEAD = WS_DATA + 4 * REGION;
constexpr size_t WS_NEED = WS_HEAD + (size_t)2 * NR * 2 * 32768;
constexpr unsigned long long DEADLINE_TICKS = 50000000ull;
constexpr unsigned long long INV_TICKS = 500ull;

struct Params {
  const u16* x; char* myws; const u16* w; u16* out;
  int rank;
  unsigned src_map;
  char* ws[NR];
};

template <int I, int E, class F> __device__ __forceinline__ void static_for(F&& f) { if constexpr (I < E) { f(std::integral_constant<int, I>{}); static_for<I + 1, E>(f); } }
__device__ __forceinline__ unsigned cvt_pk_bf16(float lo, float hi) { const f32x2 v = {lo, hi}; return __builtin_bit_cast(unsigned, __builtin_convertvector(v, bf16x2)); }
__device__ __forceinline__ unsigned rfl(unsigned v) { return __builtin_amdgcn_readfirstlane(v); }

#define SB() __builtin_amdgcn_sched_barrier(0)
#define CFENCE() asm volatile("" ::: "memory")
#define BARRIER() do { __builtin_amdgcn_s_barrier(); CFENCE(); } while (0)
#define DMA(vo, imm) do { const unsigned vo_ = (vo), so_ = soff; const i32x4 rs_ = rsrc; \
    asm volatile("buffer_load_dwordx4 %0, %1, %2 offen offset:%3 lds" :: "v"(vo_), "s"(rs_), "s"(so_), "n"(imm) : "memory"); } while (0)
#define SETM0(x) do { const unsigned m0_ = (x); asm volatile("s_mov_b32 m0, %0" :: "s"(m0_) : "memory"); } while (0)
#define WAITL0() do { CFENCE(); __builtin_amdgcn_s_waitcnt(0xC07F); } while (0)
#define WAITV(n) asm volatile("s_waitcnt vmcnt(%0)" :: "n"(n) : "memory")
#define MY_XCC() (__builtin_amdgcn_s_getreg(0xF814) & 15u)

__global__ __launch_bounds__(256, 1) void ag_gemm(Params P) {
  extern __shared__ __attribute__((aligned(16))) char smem[];
  LAS char* lds = (LAS char*)smem;
  const int wid = __builtin_amdgcn_readfirstlane(threadIdx.x >> 6);
  const int bid = blockIdx.x;
  const u16* px = P.x; char* myws = P.myws; const u16* pw = P.w; u16* pout = P.out; int rank = P.rank; unsigned src_map = P.src_map;
  asm volatile("" : "+s"(px), "+s"(myws), "+s"(pw), "+s"(pout), "+s"(rank), "+s"(src_map));

  auto whole = [&](auto ROLE) __attribute__((always_inline)) {
  constexpr bool roleW = ROLE.value;
  const int lane = threadIdx.x & 63, wh = wid & 1;
  const unsigned ldsbase = (unsigned)(size_t)lds;
  const int idx = bid >> 3, rb = idx & 1, pn = idx >> 1;
  int src_rank; bool local;
  LAS int* const src_word = (LAS int*)(lds + LDS_BYTES - SLOT);
#define GET_SOURCE() do { BARRIER(); src_rank = rfl(*src_word) & 7; local = (src_rank == rank); } while (0)
  const unsigned swz = (lane * 16) ^ ((((unsigned)lane >> 5) & 1) << 5), srow = swz >> 6, sch = (swz & 63) >> 4;
  unsigned voff[8]; i32x4 rsrc; unsigned soff = 0, sstep, m0cur, m0beg, m0end;
#define NEXT_STAGE() do { soff += sstep; m0cur += SLOT; if (m0cur == m0end) m0cur = m0beg; } while (0)
#define STAGE_ALL() do { SETM0(m0cur); asm volatile("s_nop 0"); DMA(voff[0], 0); DMA(voff[1], 1024); DMA(voff[2], 2048); DMA(voff[3], 3072); \
    SETM0(m0cur + 4096); asm volatile("s_nop 0"); DMA(voff[4], 0); DMA(voff[5], 1024); DMA(voff[6], 2048); DMA(voff[7], 3072); NEXT_STAGE(); } while (0)
#define SET_BASE(b) do { const unsigned long long b_ = (unsigned long long)(b) - 4096; rsrc.x = rfl((unsigned)b_); rsrc.y = rfl((unsigned)(b_ >> 32)); } while (0)
  constexpr int DSP = roleW ? 4 : 8, DOF = roleW ? 5 : 3, NHEAD = (32 - DOF + DSP - 1) / DSP;
#define HEAD_PIECES() do { static_for<0, NHEAD>([&](auto PP) __attribute__((always_inline)) { constexpr int pc = PP.value; \
    if constexpr ((pc & 3) == 0) { SETM0(m0cur + (pc >> 2) * 4096); asm volatile("s_nop 0"); } DMA(voff[pc], (pc & 3) * 1024); if constexpr (pc == 7) NEXT_STAGE(); }); } while (0)
  const unsigned* cnt = (const unsigned*)(myws + WS_CNT) + bid;
  unsigned call;
  f32x4 acc[8][8];
#define ZERO_ACC() do { _Pragma("unroll") for (int m = 0; m < 8; ++m) _Pragma("unroll") for (int n = 0; n < 8; ++n) { acc[m][n] = (f32x4){0.f, 0.f, 0.f, 0.f}; asm volatile("" : "+a"(acc[m][n])); } } while (0)

  auto wait_flags = [&](const bool first) __attribute__((always_inline)) {
    int ln = lane; asm volatile("" : "+v"(ln));
    const char* fp = first ? myws + WS_FIRST + (size_t)((src_rank * 4 + rb * 2 + wh) * 16 + (ln < 14 ? ln : 0)) * 128
                           : myws + WS_FLAGS + (size_t)(src_rank * 512 + rb * 256 + wh * 128 + ln * 2) * 4;
    unsigned* tmo = (unsigned*)(myws + WS_TMO);
    const unsigned* inv_done = (const unsigned*)(myws + WS_INV + MY_XCC() * 128);
    const unsigned long long t0 = __builtin_amdgcn_s_memrealtime();
    bool bad = false;
    for (;;) {
      u32x2 v; unsigned tm, dn;
      asm volatile("global_load_dwordx2 %0, %3, off sc0 sc1\n\tglobal_load_dword %1, %4, off sc1\n\tglobal_load_dword %2, %5, off sc1\n\ts_waitcnt vmcnt(0)"
                   : "=&v"(v), "=&v"(dn), "=&v"(tm) : "v"(fp), "v"(inv_done), "v"(tmo) : "memory");
      const bool ok = first ? (v.x >= call) : (((v.x >= call) && (v.y >= call)) || ln < FIRST_KT);
      const unsigned long long dt = __builtin_amdgcn_s_memrealtime() - t0;
      if (__all(ok)) {
        if (!first || rfl(dn) == call) break;
        if (dt > INV_TICKS) { asm volatile("buffer_inv sc0 sc1\n\ts_waitcnt vmcnt(0)" ::: "memory"); break; }
      } else if (rfl(tm) == call || dt > DEADLINE_TICKS) {
        asm volatile("global_store_dword %0, %1, off sc1" :: "v"(tmo), "v"(call) : "memory");
        bad = true; break;
      }
      if (first) __builtin_amdgcn_s_sleep(1); else __builtin_amdgcn_s_sleep(32);
    }
    if (bad) {
      const u32x4 nanv = {0x7fc07fc0u, 0x7fc07fc0u, 0x7fc07fc0u, 0x7fc07fc0u};
      char* nb = myws + WS_NAN + wh * 8192 + ln * 16;
      for (int i = 0; i < 8; ++i) asm volatile("global_store_dwordx4 %0, %1, off sc0 sc1\n\ts_nop 1" :: "v"(nb + i * 1024), "v"(nanv) : "memory");
      asm volatile("s_waitcnt vmcnt(0)" ::: "memory");
      SET_BASE(myws + WS_NAN); sstep = 0; soff = 0;
    }
    return bad;
  };

  i32x4 rsT; unsigned tv = 0, tso = PF_PRE << 20, tdummy = 0;
#define TOUCH(vo, so) do { const unsigned vo_ = (vo), so_ = (so); const i32x4 rs_ = rsT; \
    asm volatile("buffer_load_dword %0, %1, %2, %3 offen" : "+v"(tdummy) : "v"(vo_), "s"(rs_), "s"(so_) : "memory"); } while (0)
  int pending = 0;
  if constexpr (roleW) {
    const int piece = bid * 2 + wh;
    const int c_rb = piece >> 8, c_h = (piece >> 7) & 1, c_kt = (piece >> 1) & 63, c_q = piece & 1;
    u32x4 xv[8]; unsigned prev;
    {
      const u16* src = px + (size_t)(c_rb * 256 + (c_h * 8 + c_q * 4) * 16 + srow) * K + c_kt * 64 + sch * 8;
#pragma unroll
      for (int i = 0; i < 8; ++i)
        asm volatile("global_load_dwordx4 %0, %1, off offset:%2" : "=v"(xv[i]) : "v"(src + (size_t)(i >> 1) * 16 * K), "n"((i & 1) * 64) : "memory");
      asm volatile("global_load_dword %0, %1, off sc1" : "=v"(prev) : "v"(cnt) : "memory");
    }
    {
      const unsigned r = lane >> 3, c = lane & 7;
#pragma unroll
      for (int i = 0; i < 8; ++i) {
        const unsigned oct = wh * 2 + (i >> 2), f = ((r >> 1) & 1) | ((oct & 1) << 1), j = (c >> 1) ^ f;
        voff[i] = (oct * 8 + r) * (N * 2) + ((i & 3) * 64 + j * 16 + (c & 1) * 8) * 2 + 4096 - (i & 3) * 1024;
      }
      SET_BASE((const char*)pw + (size_t)pn * 512);
      rsrc.z = rfl((unsigned)((size_t)K * N * 2 - (size_t)pn * 512 + 4096)); rsrc.w = 0x00020000;
      sstep = 32 * N * 2; m0beg = ldsbase + wh * 8192; m0end = m0beg + NSW * SLOT; m0cur = m0beg;
    }
    unsigned long long flag_out;
    { unsigned long long wp = (unsigned long long)P.ws[0]; asm volatile("" : "+s"(wp));
#pragma unroll
      for (int p = 1; p < NR; ++p) { unsigned long long q = (unsigned long long)P.ws[p]; asm volatile("" : "+s"(q)); wp = (lane == p) ? q : wp; }
      flag_out = wp + ((c_kt >= 1 && c_kt < FIRST_KT) ? WS_FIRST + (size_t)((rank * 4 + (piece >> 7)) * 16 + 6 + (piece & 127)) * 128 : WS_FLAGS + (size_t)(rank * 512 + piece) * 4);
      asm volatile("" : "+v"(flag_out)); }
    asm volatile("s_nop 4");
    STAGE_ALL(); STAGE_ALL();
    ZERO_ACC();
    asm volatile("s_waitcnt vmcnt(16)" : "+v"(prev), "+v"(xv[0]), "+v"(xv[1]), "+v"(xv[2]), "+v"(xv[3]), "+v"(xv[4]), "+v"(xv[5]), "+v"(xv[6]), "+v"(xv[7]) :: "memory");
    call = rfl(prev) + 1u;
    {
      char* dst = myws + WS_DATA + (size_t)((call & 1u) * 2 + c_rb) * REGION + lane * 16;
#pragma unroll
      for (int i = 0; i < 8; ++i) {
        const int st = c_h * 8 + c_q * 4 + (i >> 1), ks = 2 * c_kt + (i & 1);
        asm volatile("global_store_dwordx4 %0, %1, off sc0 sc1\n\ts_nop 1" :: "v"(dst + ks * 16384 + st * 1024), "v"(xv[i]) : "memory");
      }
    }
    WAITV(0);
    if (lane < NR) asm volatile("global_store_dword %0, %1, off sc0 sc1" :: "v"(flag_out), "v"(call) : "memory");
    GET_SOURCE();
    STAGE_ALL(); STAGE_ALL();
    rsT.x = rfl((unsigned)(unsigned long long)pw); rsT.y = rfl((unsigned)((unsigned long long)pw >> 32)); rsT.z = (int)((size_t)K * N * 2); rsT.w = 0x00020000;
    asm volatile("s_nop 4");
#pragma unroll
    for (int c = 0; c < PF_PRE / 4; ++c) TOUCH(piece * 8192 + lane * 128, c << 22);
    tv = lane < 16 ? piece * 2048 + lane * 128 : 0x80000000u;
  } else {
    rsrc.z = (int)(REGION + 4096); rsrc.w = 0x00020000;
    m0beg = ldsbase + NSW * SLOT + wh * 8192; m0end = m0beg + NSX * SLOT; m0cur = m0beg;
    pending = 1;
    const int u_dst = bid & 7, u_h = idx >> 3, u_st = idx & 7;
    u32x4 xv[2]; unsigned prev;
    {
      const u16* src = px + (size_t)(u_h * 128 + u_st * 16 + srow) * K + sch * 8;
#pragma unroll
      for (int i = 0; i < 2; ++i)
        asm volatile("global_load_dwordx4 %0, %1, off offset:%2" : "=v"(xv[i]) : "v"(src), "n"(i * 64) : "memory");
    }
    if (bid < 8 && wh == 0) asm volatile("buffer_inv sc0 sc1" ::: "memory");
    asm volatile("global_load_dword %0, %1, off sc1" : "=v"(prev) : "v"(cnt) : "memory");
    char* const dstws = P.ws[u_dst];
    ZERO_ACC();
    asm volatile("s_waitcnt vmcnt(0)" : "+v"(prev), "+v"(xv[0]), "+v"(xv[1]) :: "memory");
    call = rfl(prev) + 1u;
    if (wh == 0) {
      if (bid < 8 && lane == 0) {
        unsigned* inv_done = (unsigned*)(myws + WS_INV + MY_XCC() * 128);
        asm volatile("global_store_dword %0, %1, off sc1" :: "v"(inv_done), "v"(call) : "memory");
      }
      typedef __attribute__((address_space(1))) unsigned gu32;
      gu32* claim = (gu32*)(P.myws + WS_CLAIM) + idx * 8;
      unsigned grp = MY_XCC() & 7u;
      for (int t = 0; t < 7; ++t) {
        unsigned old = 0;
        if (lane == 0) old = __hip_atomic_exchange(claim + grp, call, __ATOMIC_RELAXED, __HIP_MEMORY_SCOPE_AGENT);
        if (rfl(old) != call) break;
        grp = (grp + 1) & 7u;
      }
      if (lane == 0) *src_word = (int)((src_map >> (grp * 4)) & 7u);
      WAITL0();
    } else if (u_dst != rank) {
      char* dst = dstws + WS_HEAD + (size_t)(((call & 1u) * NR + rank) * 2 + (u_h >> 1)) * 32768 + ((u_h & 1) * 8 + u_st) * 1024 + lane * 16;
#pragma unroll
      for (int i = 0; i < 2; ++i)
        asm volatile("global_store_dwordx4 %0, %1, off sc0 sc1\n\ts_nop 1" :: "v"(dst + i * 16384), "v"(xv[i]) : "memory");
      asm volatile("s_waitcnt vmcnt(0)" ::: "memory");
      char* fl = dstws + WS_FIRST + (size_t)((rank * 4 + u_h) * 16 + u_st) * 128;
      if (lane == 0) asm volatile("global_store_dword %0, %1, off sc0 sc1" :: "v"(fl), "v"(call) : "memory");
    }
    GET_SOURCE();
    if (local) {
#pragma unroll
      for (int i = 0; i < 8; ++i) voff[i] = ((wh * 8 + i) * 16 + srow) * (K * 2) + sch * 16 + 4096 - (i & 3) * 1024;
      SET_BASE((const char*)px + (size_t)rb * REGION); sstep = 64;
    } else {
#pragma unroll
      for (int i = 0; i < 8; ++i) voff[i] = wh * 8192 + (i >> 2) * 4096 + lane * 16 + 4096;
      SET_BASE(myws + WS_HEAD + (size_t)(((call & 1u) * NR + src_rank) * 2 + rb) * 32768); sstep = 16384;
      pending = rfl(wait_flags(true) ? 0 : 1);
    }
    sstep = rfl(sstep); soff = rfl(soff);
    asm volatile("s_nop 4");
    STAGE_ALL(); STAGE_ALL();
    if (pending && !local) SET_BASE(P.ws[src_rank] + WS_DATA + (size_t)((call & 1u) * 2 + rb) * REGION);
    asm volatile("s_nop 4");
    STAGE_ALL(); STAGE_ALL();
    HEAD_PIECES();
  }

  const int wm = roleW ? 1 : 0, wn = wh, fr = lane & 15, g = lane >> 4;
  const unsigned xbase = ldsbase + NSW * SLOT + wm * 8192 + ((fr * 64 + g * 16) ^ ((fr >> 3) << 5));
  unsigned boff[4];
  { const unsigned q = (lane >> 2) & 3, p = lane & 3, f = (q >> 1) | ((g & 1) << 1);
#pragma unroll
    for (int j = 0; j < 4; ++j) { boff[j] = ldsbase + g * 4096 + wn * 2048 + q * 128 + ((j ^ f) << 5) + p * 8; asm volatile("" : "+v"(boff[j])); } }
  s16x8 Xf[4][2]; s16x4 Wf[2][8][2][2];
  unsigned xslot = 0, xcur = xbase;
#define RDX(m, kh) Xf[(m) & 3][kh] = *(const LAS s16x8*)(size_t)(xcur + (kh) * SLOT + (m) * 1024)
#define RDW(set, nb, kh, t) Wf[set][nb][kh][t] = __builtin_amdgcn_ds_read_tr16_b64_v4i16((LAS s16x4*)(size_t)(boff[(nb) & 3] + ((set) * 2 + (kh)) * SLOT + ((nb) >> 2) * 1024 + (t) * 512))
#define MFMA(set, m, n, kh) asm volatile("v_mfma_f32_16x16x32_bf16 %0, %1, %2, %0" : "+a"(acc[m][n]) : "v"(__builtin_shufflevector(Wf[set][n][kh][0], Wf[set][n][kh][1], 0, 1, 2, 3, 4, 5, 6, 7)), "v"(Xf[(m) & 3][kh]))
  if (roleW && wh == 1 && lane == 0) asm volatile("global_store_dword %0, %1, off sc1" :: "v"(cnt), "v"(call) : "memory");
#pragma unroll
  for (int n = 0; n < 8; ++n) { RDW(0, n, 0, 0); RDW(0, n, 0, 1); RDW(0, n, 1, 0); RDW(0, n, 1, 1); }
  WAITL0();
  if constexpr (!roleW) WAITV(16 + NHEAD);
  BARRIER();
  RDX(0, 0); RDX(0, 1); RDX(1, 0); RDX(1, 1);
  if constexpr (roleW) HEAD_PIECES();
  unsigned soffn = soff + sstep, m0n = m0cur + SLOT;
  if (m0n == m0end) m0n = m0beg;
  SB();

  u32x2 fv = {0u, 0u}; unsigned fd = 0;
  auto macro_step = [&](auto CS, auto CHK) __attribute__((always_inline)) {
    constexpr int cs = CS.value, ns = cs ^ 1;
    constexpr bool chk = CHK.value;
    static_for<0, 128>([&](auto I) __attribute__((always_inline)) {
      constexpr int i = I.value, m = i >> 4, n0 = (i >> 1) & 7, n = (m & 1) ? 7 - n0 : n0, kh = i & 1;
      if constexpr (i == 0) {
        if constexpr (roleW) WAITV(NHEAD + (cs == 0 ? 1 : 0));
        BARRIER();
        SB();
      }
      if constexpr (i == 96) {
        WAITL0();
        if constexpr (!roleW) WAITV(16);
        if constexpr (!roleW && chk) {
          if constexpr (cs == 0) {
            if (pending) {
              int ln = lane; asm volatile("" : "+v"(ln));
              const char* fp = myws + WS_FLAGS + (size_t)(src_rank * 512 + rb * 256 + wh * 128 + ln * 2) * 4;
              const char* dp = myws + WS_INV + MY_XCC() * 128;
              asm volatile("global_load_dwordx2 %0, %2, off sc0 sc1\n\tglobal_load_dword %1, %3, off sc1" : "+v"(fv), "+v"(fd) : "v"(fp), "v"(dp) : "memory");
            }
          } else {
            if (pending) {
              asm volatile("" : "+v"(fv), "+v"(fd));
              const bool all = __all(((fv.x >= call) && (fv.y >= call)) || lane < FIRST_KT);
              if (local) {
                if (all && rfl(fd) == call) {
#pragma unroll
                  for (int q = 0; q < 8; ++q) voff[q] = wh * 8192 + (q >> 2) * 4096 + lane * 16 + 4096;
                  SET_BASE(myws + WS_DATA + (size_t)((call & 1u) * 2 + rb) * REGION); sstep = 16384; soff = 2 * FIRST_KT * 16384;
                }
              } else if (!all) wait_flags(false);
              sstep = rfl(sstep); soff = rfl(soff);
              pending = 0;
              WAITL0();
              asm volatile("s_nop 4");
            }
          }
        }
        BARRIER();
        SB();
      }
      MFMA(cs, m, n, kh);
      if constexpr ((i & 15) == 0) RDX((m + 2) & 7, 0);
      if constexpr ((i & 15) == 2) RDX((m + 2) & 7, 1);
      if constexpr (i == 88) { xslot += 2 * SLOT; asm volatile("" : "+s"(xslot)); }
      if constexpr (i == 90) { if (xslot == NSX * SLOT) xslot = 0; asm volatile("" : "+s"(xslot)); }
      if constexpr (i == 92) { xcur = xbase + xslot; asm volatile("" : "+v"(xcur)); }
      if constexpr (i < 92) {
        constexpr int pos = i & 15, sl = (pos >= 4 && (pos & 1) == 0) ? (pos - 4) / 2 : -1;
        if constexpr (sl >= 0 && m * 6 + sl < 32) { constexpr int r = m * 6 + sl; RDW(ns, r >> 2, (r >> 1) & 1, r & 1); }
      }
      if constexpr (roleW && cs == 1 && i == 16 * DSP + DOF - 32) TOUCH(tv, tso);
      if constexpr (roleW && cs == 1 && i == 16 * DSP + DOF - 30) { tso += 1u << 20; asm volatile("" : "+s"(tso)); }
      constexpr int d = ((i + 32) & 127) - DOF;
      if constexpr (d + 2 >= 0 && (d + 2) % (4 * DSP) == 0 && (d + 2) / DSP < 16) SETM0(m0cur + ((((d + 2) / DSP) >> 2) & 1) * 4096);
      if constexpr (d >= 0 && d % DSP == 0 && d / DSP < 16) {
        constexpr int piece = (d / DSP) & 7;
        DMA(voff[piece], (piece & 3) * 1024);
        if constexpr (piece == 7) { soff = soffn; m0cur = m0n; }
      }
      if constexpr (d >= 2 && (d - 2) % DSP == 0 && (d - 2) / DSP < 16) {
        constexpr int piece = ((d - 2) / DSP) & 7;
        if constexpr (piece == 0) { soffn = soff + sstep; asm volatile("" : "+s"(soffn)); }
        if constexpr (piece == 1) { m0n = m0cur + SLOT; asm volatile("" : "+s"(m0n)); }
        if constexpr (piece == 2) { if (m0n == m0end) m0n = m0beg; asm volatile("" : "+s"(m0n)); }
      }
      SB();
    });
  };
  if constexpr (!roleW) {
    macro_step(std::integral_constant<int, 0>{}, std::true_type{});
    macro_step(std::integral_constant<int, 1>{}, std::true_type{});
  }
  for (int j = roleW ? 0 : 2; j < NJ - 2; j += 2) {
    macro_step(std::integral_constant<int, 0>{}, std::false_type{});
    macro_step(std::integral_constant<int, 1>{}, std::false_type{});
  }
#define ACC_RD(m, n, r) ({ float t_; asm volatile("v_accvgpr_read_b32 %0, %1" : "=v"(t_) : "a"(acc[m][n][r])); t_; })
  u16* outp = pout + (size_t)((src_rank * 2 + rb) * 256 + wm * 128 + fr) * N + pn * 256 + wn * 128 + (g & 1) * 16 + (g >> 1) * 8;
  unsigned d0, d1, e0, e1;
  auto emit = [&](auto MM, auto JJ, auto QQ) __attribute__((always_inline)) {
    constexpr int m = MM.value, j = JJ.value, q = QQ.value;
    if constexpr (q == 0) { d0 = cvt_pk_bf16(ACC_RD(m, 2 * j, 0), ACC_RD(m, 2 * j, 1)); d1 = cvt_pk_bf16(ACC_RD(m, 2 * j, 2), ACC_RD(m, 2 * j, 3)); }
    if constexpr (q == 1) { e0 = cvt_pk_bf16(ACC_RD(m, 2 * j + 1, 0), ACC_RD(m, 2 * j + 1, 1)); e1 = cvt_pk_bf16(ACC_RD(m, 2 * j + 1, 2), ACC_RD(m, 2 * j + 1, 3)); }
    if constexpr (q == 2) { auto r0 = __builtin_amdgcn_permlane16_swap(d0, e0, false, false); auto r1 = __builtin_amdgcn_permlane16_swap(d1, e1, false, false);
                            d0 = r0[0]; e0 = r0[1]; d1 = r1[0]; e1 = r1[1]; }
    if constexpr (q == 3) { const u32x4 w = {d0, d1, e0, e1}; u16* const op = outp + (size_t)m * 16 * N;
      asm volatile("global_store_dwordx4 %0, %1, off offset:%2 sc1\n\ts_nop 1" :: "v"(op), "v"(w), "n"(j * 64) : "memory"); }
  };
  xslot += 2 * SLOT; if (xslot == NSX * SLOT) xslot = 0;
  unsigned xnext = xbase + xslot; asm volatile("" : "+v"(xnext));
#define RDXS(xs, xb, m) do { Xf[xs][0] = *(const LAS s16x8*)(size_t)((xb) + (m) * 1024); Xf[xs][1] = *(const LAS s16x8*)(size_t)((xb) + SLOT + (m) * 1024); } while (0)
#define MFMX(set, m, n, kh, xs) asm volatile("v_mfma_f32_16x16x32_bf16 %0, %1, %2, %0" : "+a"(acc[m][n]) : "v"(__builtin_shufflevector(Wf[set][n][kh][0], Wf[set][n][kh][1], 0, 1, 2, 3, 4, 5, 6, 7)), "v"(Xf[xs][kh]))
  static_for<0, 256>([&](auto I) __attribute__((always_inline)) {
    constexpr int i = I.value;
    if constexpr (i == 0) {
      if constexpr (roleW) asm volatile("s_waitcnt vmcnt(%0)" :: "n"(NHEAD), "v"(tdummy) : "memory");
      BARRIER();
      SB();
    }
    if constexpr (i == 48) {
      if constexpr (!roleW) WAITV(NHEAD);
      BARRIER();
      SB();
    }
    if constexpr (i < 64) {
      constexpr int m = i >> 4, n = (i >> 1) & 7, kh = i & 1, pos = i & 15;
      MFMX(0, m, n, kh, m);
      if constexpr (i == 0) RDXS(2, xcur, 2);
      if constexpr (i == 16) RDXS(3, xcur, 3);
      if constexpr (i == 32) RDXS(0, xcur, 4);
      if constexpr (i == 48) RDXS(1, xnext, 4);
      if constexpr (pos >= 3 && pos < 11) { constexpr int r = m * 8 + pos - 3; RDW(1, r >> 2, (r >> 1) & 1, r & 1); }
    } else if constexpr (i < 192) {
      constexpr int t = i - 64, m = 4 + (t >> 5), n = (t >> 2) & 7, set = (t >> 1) & 1, kh = t & 1, xs = (m & 1) * 2 + set, pos = t & 31;
      MFMX(set, m, n, kh, xs);
      if constexpr (m < 7) {
        if constexpr (pos == 0) RDXS(((m + 1) & 1) * 2, xcur, m + 1);
        if constexpr (pos == 4) RDXS(((m + 1) & 1) * 2 + 1, xnext, m + 1);
      } else {
        if constexpr (pos == 0) RDXS(0, xnext, 0);
        if constexpr (pos == 4) RDXS(1, xnext, 1);
      }
      if constexpr (m > 4 && (pos & 1) == 1) emit(std::integral_constant<int, m - 1>{}, std::integral_constant<int, (pos >> 3)>{}, std::integral_constant<int, ((pos >> 1) & 3)>{});
    } else {
      constexpr int t = i - 192, m = t >> 4, n = (t >> 1) & 7, kh = t & 1;
      MFMX(1, m, n, kh, m);
      if constexpr (t == 0) RDXS(2, xnext, 2);
      if constexpr (t == 2) RDXS(3, xnext, 3);
      emit(std::integral_constant<int, (m == 0 ? 7 : m - 1)>{}, std::integral_constant<int, ((t >> 2) & 3)>{}, std::integral_constant<int, (t & 3)>{});
    }
    SB();
  });
  static_for<0, 16>([&](auto I) __attribute__((always_inline)) {
    constexpr int i = I.value;
    emit(std::integral_constant<int, 3>{}, std::integral_constant<int, (i >> 2)>{}, std::integral_constant<int, (i & 3)>{});
    SB();
  });
  };
  if (wid >> 1) whole(std::true_type{}); else whole(std::false_type{});
}

extern "C" void kernel_launch(int rank, int nranks, ncclComm_t comm,
                              hipStream_t stream,
                              void* const* d_in, const int* in_sizes, int n_in,
                              void* d_out, int out_size,
                              const int* in_dtypes, int out_dtype,
                              void* const* d_out_all,
                              void* const* d_ws_all, size_t ws_bytes) {
  if (nranks != NR || (unsigned)rank >= (unsigned)NR || n_in < 2 || ws_bytes < WS_NEED || in_sizes[0] != 512 * K || in_sizes[1] != K * N || out_size != M * N) {
    fprintf(stderr, "ag_gemm: unexpected problem shape\n");
    return;
  }
  static bool attr_set = false;
  if (!attr_set) {
    HIP_CHECK(hipFuncSetAttribute((const void*)ag_gemm, hipFuncAttributeMaxDynamicSharedMemorySize, LDS_BYTES));
    attr_set = true;
  }
  Params P;
  P.x = (const u16*)d_in[0]; P.w = (const u16*)d_in[1]; P.out = (u16*)d_out; P.rank = rank; P.myws = (char*)d_ws_all[rank];
  for (int p = 0; p < NR; ++p) P.ws[p] = (char*)d_ws_all[p];
  {
    static const int half_of[NR] = {0, 1, 0, 0, 1, 1, 0, 1};
    constexpr int SELF_XCD = 3;
    int map[NR], lo = 0, hi = 4;
    for (int j = 1; j < NR; ++j) {
      const int p = (rank + j) % NR;
      if (half_of[p] == half_of[rank]) { if (lo == SELF_XCD) ++lo; map[lo++] = p; } else map[hi++] = p;
    }
    map[SELF_XCD] = rank;
    P.src_map = 0;
    for (int g = 0; g < NR; ++g) P.src_map |= (unsigned)map[g] << (4 * g);
  }
  hipLaunchKernelGGL(ag_gemm, dim3(256), dim3(256), LDS_BYTES, stream, P);
  HIP_CHECK(hipGetLastError());
}
```
